```python
import numpy as np

kernel_name = "gpu_gemm_rs_m4096_k4096_n32768_f32_gelu"


ABSMAX_RTOL = 0.001

COMPUTE_DTYPE = "float32"
EPILOGUE = "gelu"
IN_QUANT = None
OUT_QUANT = None
IN_DTYPES = {"x": "float32", "w_mat": "float32"}
WORLD_SIZE = 8

def setup_inputs():

    x = np.random.randn(4096, 4096).astype(np.float32)
    w_mat = (np.random.randn(4096, 32768) / np.sqrt(4096)).astype(np.float32)
    return {"x": x, "w_mat": w_mat}

def _epilogue(y):
    c = 0.7978845608028654
    return (0.5 * y * (1.0 + np.tanh(c * (y + 0.044715 * y ** 3)))).astype(np.float32)

def reference(x, w_mat):
    y = (x @ w_mat).astype(np.float32)
    return _epilogue(y)
```

```cpp
#include <hip/hip_runtime.h>
#include <rccl/rccl.h>
#include <cstdio>
#include <cstdint>

#define HIP_CHECK(expr) do { hipError_t _e = (expr); if (_e != hipSuccess) { \
    fprintf(stderr, "HIP error %s at %s:%d\n", hipGetErrorString(_e), __FILE__, __LINE__); \
    return; } } while (0)

#define FIN1 14
#define FIN2 40
#define FINX 45
#define FIN3 95
#define NTOUT 1
#define TAIL2 24u
#define SOLO 80u
#define SPREAD 1
#define AHEAD_IN 4
#define AHEAD_Y 3
#define PWIN 1
#define GATE 0
#define CRIT 80
#define PACE 13

namespace gk {
#define LAS __attribute__((address_space(3)))
#define GAS __attribute__((address_space(1)))
typedef unsigned u32;
typedef unsigned long long u64;
typedef u32 u32x2 __attribute__((ext_vector_type(2)));
typedef u32 u32x3 __attribute__((ext_vector_type(3)));
typedef u32 u32x4 __attribute__((ext_vector_type(4)));
typedef float f32x4 __attribute__((ext_vector_type(4)));
typedef _Float16 h16x8 __attribute__((ext_vector_type(8)));
typedef _Float16 h16x2 __attribute__((ext_vector_type(2)));

constexpr int NR = 8;
constexpr int MM = 4096, KS = 512, NN = 32768, NB = 4096;
constexpr int NWG = 256, NCVT = 144;
constexpr int NCONV = 1152;
constexpr u64 DEADLINE = 40000000ull;

constexpr size_t MiB = 1ull << 20;
constexpr size_t PIECE = 32768;
constexpr int PL_IN = 3328, PL_Y = 6656;
constexpr int UB_IN = 1024 + 2 * PL_IN, UB_Y = 1024 + 2 * PL_Y;
constexpr u32 TK_IN = 256 * 4, TK_SOLO = TK_IN + 32 * 16 - SOLO, TK_END = TK_SOLO + 2 * SOLO;
constexpr u32 TK_0 = 8 * 4;
constexpr u32 TICKS = TK_END - TK_0 + 64;
constexpr int C_CALL = 0;
constexpr int C_TMO = 8192;
constexpr int C_R2 = 8192 + 160;
constexpr int C_TICK = 8192 + 288;
constexpr int C_YSLOT = 8192 + 512;
constexpr int C_FLAGA = 9216;
constexpr int C_FLAGB = 10240;
constexpr int C_RDY = 11264;
constexpr int C_YQ = 13824;
constexpr int C_END = 14080;
constexpr size_t OFF_STAMP = 256 * 1024;
constexpr size_t OFF_STA = 1 * MiB;
constexpr size_t OFF_STB = 33 * MiB;
constexpr size_t OFF_EXX = 65 * MiB;
constexpr size_t OFF_EXW = 73 * MiB;
constexpr size_t OFF_EXY = 137 * MiB;
constexpr size_t OFF_RWX = 265 * MiB;
constexpr size_t OFF_RWW = 273 * MiB;
constexpr size_t OFF_RWY = 337 * MiB;
constexpr size_t OFF_PSH = 465 * MiB;
constexpr size_t WS_NEED = 469 * MiB;
static_assert(C_END * 4 <= OFF_STAMP && 4 * UB_IN <= PIECE && UB_Y <= 16384, "layout");

constexpr int L_STAGE = 0;
constexpr int L_SCR = 131072;
constexpr int L_MISC = L_SCR + 1024;
constexpr int L_LUT = L_MISC + 256;
constexpr int L_XT = L_LUT + 128;
constexpr int L_VAR = L_XT + 16384;
constexpr int LDS_BYTES = L_VAR + 8 * 1040;

struct Order { u32 item[256]; u32 place[256]; u32 conv[1152]; };
constexpr Order make_order() {
    Order o{};
    int key[256] = {};
    for (int i = 0; i < 256; ++i) {
        const int kk = i >> 5, r = i & 31; const bool isx = r < 16;
        const int t = isx ? (r & 7) * 2 + (r >> 3) : (r & 3) * 4 + ((r >> 2) & 3);
        constexpr int fin_w[4] = {0, FIN1, FIN2, FIN3};
        const int k = (isx ? ((t & 1) ? FINX : 0) : fin_w[t & 3]) + 10 * kk;
        int j = i;
        for (; j > 0 && key[j - 1] > k; --j) { key[j] = key[j - 1]; o.item[j] = o.item[j - 1]; }
        key[j] = k; o.item[j] = (u32)((isx ? 128 : 0) | kk << 4 | t);
    }
    int n = 0;
    for (int i = 0; i < 256; ++i) {
        o.place[o.item[i]] = (u32)i;
        for (int d = 0; d < ((o.item[i] & 128u) ? 1 : 8); ++d) o.conv[n++] = o.item[i] | (u32)d << 8;
    }
    return o;
}
__constant__ const Order ORDER = make_order();
static_assert(make_order().item[0] == 128 && make_order().item[7] == 142 && make_order().conv[7] == 142, "the pushed pieces are x, k block 0, even row tiles");

struct Params {
    const float* x; const float* w; float* out; char* self;
    int rank, pad;
    char* ws[NR];
    u32 first[CRIT];
};

#define WAIT_V(n) asm volatile("s_waitcnt vmcnt(" #n ")" ::: "memory")
#define WAIT_L(n) asm volatile("s_waitcnt lgkmcnt(" #n ")" ::: "memory")
#define WAIT_V0_SEEN() do { __builtin_amdgcn_s_waitcnt(0x0F70); asm volatile("" ::: "memory"); } while (0)
#define BAR __builtin_amdgcn_s_barrier()
#define SCHED __builtin_amdgcn_sched_barrier(0)

__device__ __forceinline__ void st16_sys(char* p, u32x4 v) { asm volatile("global_store_dwordx4 %0, %1, off sc0 sc1\n\ts_nop 1" :: "v"(p), "v"(v) : "memory"); }
__device__ __forceinline__ void st16_nt(char* p, u32x4 v) { asm volatile("global_store_dwordx4 %0, %1, off sc0 sc1 nt\n\ts_nop 1" :: "v"(p), "v"(v) : "memory"); }
__device__ __forceinline__ void st8_sys(char* p, u32x2 v) { asm volatile("global_store_dwordx2 %0, %1, off sc0 sc1" :: "v"(p), "v"(v) : "memory"); }
__device__ __forceinline__ void st4_sys(char* p, u32 v) { asm volatile("global_store_dword %0, %1, off sc0 sc1" :: "v"(p), "v"(v) : "memory"); }
__device__ __forceinline__ u32 ld_sys(const u32* p) { return __hip_atomic_load(p, __ATOMIC_RELAXED, __HIP_MEMORY_SCOPE_SYSTEM); }
__device__ __forceinline__ u32 ld_agt(const u32* p) { return __hip_atomic_load(p, __ATOMIC_RELAXED, __HIP_MEMORY_SCOPE_AGENT); }
__device__ __forceinline__ void st_sys(u32* p, u32 v) { __hip_atomic_store(p, v, __ATOMIC_RELAXED, __HIP_MEMORY_SCOPE_SYSTEM); }
__device__ __forceinline__ void st_agt(u32* p, u32 v) { __hip_atomic_store(p, v, __ATOMIC_RELAXED, __HIP_MEMORY_SCOPE_AGENT); }
__device__ __forceinline__ u32 add_agt(u32* p, u32 v) { return __hip_atomic_fetch_add(p, v, __ATOMIC_RELAXED, __HIP_MEMORY_SCOPE_AGENT); }
__device__ __forceinline__ u32 rfl(u32 v) { return (u32)__builtin_amdgcn_readfirstlane((int)v); }
__device__ __forceinline__ u32 pk16(float a, float b) { h16x2 h; h.x = (_Float16)a; h.y = (_Float16)b; return __builtin_bit_cast(u32, h); }
__device__ __forceinline__ float gelu(float y) {
    const float c1 = -2.0f * 0.7978845608028654f * 1.4426950408889634f, c3 = c1 * 0.044715f;
    const float e = __builtin_amdgcn_exp2f(y * __builtin_fmaf(y * y, c3, c1));
    return y * __builtin_amdgcn_rcpf(1.0f + e);
}
constexpr int VAR = 1024;
constexpr int VAR0 = 384;
__device__ __forceinline__ u32 perm(u32 hi, u32 lo, u32 sel) { return __builtin_amdgcn_perm(hi, lo, sel); }
__device__ __forceinline__ u32 dot4(u32 a, u32 b, u32 c) { return __builtin_amdgcn_udot4(a, b, c, false); }
__device__ __forceinline__ u32 funnel(u32 hi, u32 lo, u32 s) { return __builtin_amdgcn_alignbit(hi, lo, s); }
__device__ __forceinline__ u32 last_lane(u32 v) { return (u32)__builtin_amdgcn_readlane((int)v, 63); }
__device__ __forceinline__ u32 wave_scan(u32 v) {
    v += (u32)__builtin_amdgcn_update_dpp(0, (int)v, 0x111, 0xf, 0xf, false);
    v += (u32)__builtin_amdgcn_update_dpp(0, (int)v, 0x112, 0xf, 0xf, false);
    v += (u32)__builtin_amdgcn_update_dpp(0, (int)v, 0x114, 0xf, 0xf, false);
    v += (u32)__builtin_amdgcn_update_dpp(0, (int)v, 0x118, 0xf, 0xf, false);
    v += (u32)__builtin_amdgcn_update_dpp(0, (int)v, 0x142, 0xa, 0xf, false);
    v += (u32)__builtin_amdgcn_update_dpp(0, (int)v, 0x143, 0xc, 0xf, false);
    return v;
}
__device__ __forceinline__ void make_lut(LAS u32* lut, int m) {
    u32 spread = 0, gather = 0x0c0c0c0cu; int n = 0;
    for (int b = 0; b < 4; ++b) {
        if ((m >> b) & 1) { spread |= (u32)n << (8 * b); gather = (gather & ~(0xffu << (8 * n))) | (u32)b << (8 * n); ++n; }
        else spread |= 0x0cu << (8 * b);
    }
    lut[m] = spread; lut[16 + m] = gather;
}
__device__ __forceinline__ void or_lds(LAS unsigned char* p, u32 v) { __hip_atomic_fetch_or((LAS u32*)p, v, __ATOMIC_RELAXED, __HIP_MEMORY_SCOPE_WAVEFRONT); }

__device__ __forceinline__ void or_bits(LAS unsigned char* var, u32 at, u32 v) {
    LAS unsigned char* const p = var + (at >> 5) * 4;
    or_lds(p, v << (at & 31u)); or_lds(p + 4, (v >> 1) >> (31u - (at & 31u)));
}
__device__ __forceinline__ u32 get_bits(const LAS unsigned char* var, u32 at) {
    const volatile LAS u32* const p = (const volatile LAS u32*)(var + (at >> 5) * 4);
    const u32 lo = p[0], hi = p[1];
    return funnel(hi, lo, at);
}

struct ExpCode { u32 cc[2], s2[2], len[2], m3; int e0; };
__device__ __forceinline__ u32 both_bits(const u32 (&w)[2]) { return (w[0] & (w[0] >> 1) & 0x55555555u) | (w[1] & (w[1] << 1) & 0xaaaaaaaau); }
__device__ __forceinline__ u32 value_of(u32 bit) { return ((bit & 7u) << 2) | (bit >> 3); }
template <bool F32, class Hook>
__device__ __forceinline__ void code_frame(const u32 (&E)[8], u32 lut, ExpCode& x, Hook step) {
    u32 esum = 0;
#pragma unroll
    for (int q = 0; q < 8; ++q) esum = __builtin_amdgcn_sad_u8(E[q], 0u, esum);
    const int m = ((int)last_lane(wave_scan(esum)) - 124) >> 11;
    x.e0 = m < 2 ? 2 : (m > (F32 ? 252 : 28) ? (F32 ? 252 : 28) : m);
    const u32 lo = (u32)(x.e0 - 2) * 0x01010101u, k1 = lo & 0x7f7f7f7fu, k2 = ~lo & 0x80808080u;
    u32 dd[2] = {0, 0};
    x.cc[0] = x.cc[1] = x.s2[0] = x.s2[1] = x.len[0] = x.len[1] = 0;
#pragma unroll
    for (int q = 0; q < 8; ++q) {
        const u32 u = F32 ? ((E[q] | 0x80808080u) - k1) ^ ((E[q] & 0x80808080u) ^ k2) : ((E[q] | 0x80808080u) - lo) ^ 0x80808080u;
        const u32 big = (((u >> 3) & 0x1f1f1f1fu) + 0x1f1f1f1fu) & 0x20202020u;
        const u32 idx = (u & 0x07070707u) | ((big >> 2) - (big >> 5));
        const u32 c = perm(0x03030302u, 0x01000303u, idx), d = perm(0x03030200u, 0x00000001u, idx), f = perm(0x01010100u, 0x00000101u, idx);
        const u32 g = perm(0u, d, *(const LAS u32*)(dot4(f, 0x20100804u, lut) + 64u));
        const u32 h = g | (g >> 6);
        x.cc[q & 1] |= c << (q & 6);
        dd[q & 1] |= d << (q & 6);
        x.s2[q & 1] |= ((h & 0x0fu) | ((h >> 12) & 0xf0u)) << x.len[q & 1];
        x.len[q & 1] = dot4(f, 0x02020202u, x.len[q & 1]);
        step(q);
    }
    x.m3 = both_bits(dd);
}
template <bool F32>
__device__ __forceinline__ u32 put_var(LAS unsigned char* sc, ExpCode (&x)[2], char* unit, int lane) {
    LAS unsigned char* const var = sc + 4096;
    *(volatile LAS u32x4*)(var + lane * 16) = (u32x4){0, 0, 0, 0};
    u32 p[2], t[2], s3[2] = {0, 0}, mx[2] = {0, 0}, odd = 0;
#pragma unroll
    for (int f = 0; f < 2; ++f) {
        const u32 n3 = (u32)__builtin_popcount(x[f].m3), n = (x[f].len[0] + x[f].len[1]) | n3 << 16, inc = wave_scan(n);
        p[f] = inc - n; t[f] = last_lane(inc);
        if (n3 > 10u) odd = 4u;
    }
    u32 m[2] = {x[0].m3, x[1].m3};
    for (u32 sh = 0; sh < 30u && (m[0] | m[1]); sh += 3u) {
#pragma unroll
        for (int f = 0; f < 2; ++f) {
            const u32 v = value_of((u32)__builtin_ctz(m[f] | 0x80000000u)), code = (u32)(x[f].e0 - 3 - *(volatile LAS unsigned char*)(sc + f * 2048 + lane * 32 + v));
            const u32 bit = m[f] & (0u - m[f]);
            s3[f] |= (code > 7u || bit == 0u) ? 0u : code << sh;
            mx[f] |= code > 7u ? bit : 0u;
            m[f] ^= bit;
        }
    }
    const u32 nx = (u32)__builtin_popcount(mx[0]) | (u32)__builtin_popcount(mx[1]) << 16, incx = wave_scan(nx);
    u32 tx = last_lane(incx);
    const u32 b3 = 96u + (t[0] & 0xffffu) + (t[1] & 0xffffu), bx = (b3 + 3u * ((t[0] >> 16) + (t[1] >> 16)) + 15u) >> 4;
    const u32 bytes = (bx + (tx & 0xffffu) + (tx >> 16)) * 2u;
    u32 raw = 3u;
    if ((tx & 0xffffu) <= 64u && (tx >> 16) <= 64u && bytes <= (u32)VAR && __ballot(odd) == 0ull) {
#pragma unroll
        for (int f = 0; f < 2; ++f) {
            const u32 o2 = 96u + (p[f] & 0xffffu) + (f ? t[0] & 0xffffu : 0u);
            or_bits(var, o2, x[f].s2[0]); or_bits(var, o2 + x[f].len[0], x[f].s2[1]);
            or_bits(var, b3 + 3u * ((p[f] >> 16) + (f ? t[0] >> 16 : 0u)), s3[f]);
        }
        if (tx) {
#pragma unroll
            for (int f = 0; f < 2; ++f) {
                u32 ox = bx + (f ? ((incx - nx) >> 16) + (tx & 0xffffu) : (incx - nx) & 0xffffu);
                for (; mx[f]; mx[f] &= mx[f] - 1u) {
                    const int v = (int)value_of((u32)__builtin_ctz(mx[f])), e = *(volatile LAS unsigned char*)(sc + f * 2048 + lane * 32 + v), e0 = x[f].e0;
                    int code = e;
                    if (F32) {
                        code = e < e0 ? e0 - 11 - e : e - e0 + 20;
                        if (code > (e < e0 ? 23 : 30)) { code = 31; if (e != 0) odd |= 1u << f; }
                    }
                    *(volatile LAS unsigned short*)(var + ox * 2) = (unsigned short)((lane * 32 + v) | code << 11);
                    ++ox;
                }
            }
        }
        raw = (__ballot(odd & 1u) ? 1u : 0u) | (__ballot(odd & 2u) ? 2u : 0u);
    } else
    { tx = t[0] = t[1] = 0u; }
    if (lane == 0) *(volatile LAS u32x3*)var = (u32x3){(u32)x[0].e0 | (u32)x[1].e0 << 8 | (tx & 0xffffu) << 16 | (tx >> 16) << 23 | raw << 30,
                                                       (t[0] & 0xffffu) | t[1] << 16, t[0] >> 16 | (t[1] & 0xffff0000u)};
    if ((u32)lane * 16u < (raw == 3u ? 12u : bytes)) st16_sys(unit + lane * 16, *(volatile LAS u32x4*)(var + lane * 16));
    return raw;
}
__device__ __forceinline__ u32 var_list(u32 h1, u32 h2) { return (96u + (h1 & 0xffffu) + (h1 >> 16) + 3u * ((h2 & 0xffffu) + (h2 >> 16)) + 15u) >> 4; }
__device__ __forceinline__ u32 var_bytes(u32 h0, u32 h1, u32 h2) { return (var_list(h1, h2) + ((h0 >> 16) & 127u) + ((h0 >> 23) & 127u)) * 2u; }
template <bool F32>
__device__ __forceinline__ void decode_exp(const LAS unsigned char* var, u32 h0, u32 h1, u32 h2, int f, const u32 (&cc)[2], u32 lut, LAS unsigned char* xt,
                                           u32 (&E)[8], int lane) {
    constexpr int SH = F32 ? 0 : 2;
    const int e0 = (h0 >> (8 * f)) & 255u;
    const u32 more = both_bits(cc);
    const u32 la = 2u * (u32)__builtin_popcount(more & 0x55555555u), l2 = 2u * (u32)__builtin_popcount(more);
    const u32 o2 = 96u + wave_scan(l2) - l2 + (f ? h1 & 0xffffu : 0u);
    const u32 s2[2] = {get_bits(var, o2), get_bits(var, o2 + la)};
    const u32 tlo = ((u32)e0 * 0x01010101u + 0x00020100u - 0x01000000u) << SH, thi = ((u32)(e0 - 2) | (u32)(e0 + 3) << 8 | (u32)e0 << 16) << SH;
    u32 off[2] = {0, 0}, dd[2] = {0, 0};
#pragma unroll
    for (int q = 0; q < 8; ++q) {
        const u32 fl = (more >> q) & 0x01010101u;
        const u32 w = s2[q & 1] >> off[q & 1];
        const u32 u = (w & 0x0fu) | ((w << 12) & 0x000f0000u);
        const u32 d = perm(0u, (u | (u << 6)) & 0x03030303u, *(const LAS u32*)dot4(fl, 0x20100804u, lut));
        off[q & 1] = dot4(fl, 0x02020202u, off[q & 1]);
        E[q] = perm(thi, tlo, ((cc[q & 1] >> (q & 6)) & 0x03030303u) + d);
        dd[q & 1] |= d << (q & 6);
    }
    u32 m3 = both_bits(dd);
    const u32 n3 = (u32)__builtin_popcount(m3);
    u32 s3 = get_bits(var, 96u + (h1 & 0xffffu) + (h1 >> 16) + 3u * (wave_scan(n3) - n3 + (f ? h2 & 0xffffu : 0u)));
    for (; m3; m3 &= m3 - 1u, s3 >>= 3)
        *(volatile LAS unsigned char*)(xt + lane * 32 + value_of((u32)__builtin_ctz(m3))) = (unsigned char)((e0 ^ (e0 - 3 - (int)(s3 & 7u))) << SH);
    if ((u32)lane < ((h0 >> (f ? 23 : 16)) & 127u)) {
        const int ent = *(const volatile LAS unsigned short*)(var + (var_list(h1, h2) + (f ? (h0 >> 16) & 127u : 0u) + lane) * 2), code = ent >> 11;
        *(volatile LAS unsigned char*)(xt + (ent & 2047)) = (unsigned char)(((F32 ? (code < 24 ? e0 - 11 - code : (code < 31 ? e0 - 20 + code : 0)) : code) ^ e0) << SH);
    }
    const u32x4 a = *(volatile LAS u32x4*)(xt + lane * 32), b = *(volatile LAS u32x4*)(xt + lane * 32 + 16);
    *(volatile LAS u32x4*)(xt + lane * 32) = (u32x4){0, 0, 0, 0}; *(volatile LAS u32x4*)(xt + lane * 32 + 16) = (u32x4){0, 0, 0, 0};
    E[0] ^= a.x; E[1] ^= a.y; E[2] ^= a.z; E[3] ^= a.w; E[4] ^= b.x; E[5] ^= b.y; E[6] ^= b.z; E[7] ^= b.w;
}

__device__ __forceinline__ void pack_in(LAS unsigned char* sc, u32 lut, const u32x4 (&c)[2][4], char* unit, char* rawdst, int lane) {
    ExpCode x[2];
#pragma unroll
    for (int f = 0; f < 2; ++f) {
        u32 L[8], E[8], mm[2] = {0, 0}, sg = 0;
#pragma unroll
        for (int i = 0; i < 8; ++i) {
            const u32 a = c[f][i >> 1][(i & 1) * 2], b = c[f][i >> 1][(i & 1) * 2 + 1], h = perm(b, a, 0x07050301u);
            L[i] = perm(b, a, 0x06040200u);
            E[i] = (h >> 2) & 0x1f1f1f1fu;
            mm[i >> 2] |= (h & 0x03030303u) << (2 * (i & 3));
            sg |= (h & 0x80808080u) >> (7 - i);
        }
        char* const dst = unit + VAR + f * PL_IN;
        st16_sys(dst + lane * 16, (u32x4){L[0], L[1], L[2], L[3]});
        st16_sys(dst + 1024 + lane * 16, (u32x4){L[4], L[5], L[6], L[7]});
        *(volatile LAS u32x4*)(sc + f * 2048 + lane * 32) = (u32x4){E[0], E[1], E[2], E[3]};
        *(volatile LAS u32x4*)(sc + f * 2048 + lane * 32 + 16) = (u32x4){E[4], E[5], E[6], E[7]};
        code_frame<false>(E, lut, x[f], [](int) {});
        st16_sys(dst + 2048 + lane * 16, (u32x4){x[f].cc[0], x[f].cc[1], mm[0], mm[1]});
        st4_sys(dst + 3072 + lane * 4, sg);
    }
    const u32 raw = put_var<false>(sc, x, unit, lane);
#pragma unroll
    for (int f = 0; f < 2; ++f)
        if ((raw >> f) & 1u) {
#pragma unroll
            for (int q = 0; q < 4; ++q) st16_sys(rawdst + f * 4096 + q * 1024 + lane * 16, c[f][q]);
        }
}
template <class Hook>
__device__ __forceinline__ void unpack_in(const LAS unsigned char* pl, const LAS unsigned char* var, u32 h0, u32 h1, u32 h2, int f, u32 lut, LAS unsigned char* xt,
                                          u32x4 (&c)[4], int lane, Hook mid) {
    const u32x4 k = *(const volatile LAS u32x4*)(pl + 2048 + lane * 16);
    const u32 cc[2] = {k.x, k.y}, mm[2] = {k.z, k.w}, sg = *(const volatile LAS u32*)(pl + 3072 + lane * 4);
    u32 E[8];
    decode_exp<false>(var, h0, h1, h2, f, cc, lut, xt, E, lane);
    mid();
    const u32x4 l0 = *(const volatile LAS u32x4*)(pl + lane * 16), l1 = *(const volatile LAS u32x4*)(pl + 1024 + lane * 16);
    const u32 L[8] = {l0.x, l0.y, l0.z, l0.w, l1.x, l1.y, l1.z, l1.w};
#pragma unroll
    for (int i = 0; i < 8; ++i) {
        const u32 h = (E[i] & 0x7c7c7c7cu) | ((mm[i >> 2] >> (2 * (i & 3))) & 0x03030303u) | ((sg << (7 - i)) & 0x80808080u);
        c[i >> 1][(i & 1) * 2] = perm(h, L[i], 0x05010400u);
        c[i >> 1][(i & 1) * 2 + 1] = perm(h, L[i], 0x07030602u);
    }
}
__device__ __forceinline__ void pack_y(LAS unsigned char* sc, u32 lut, const f32x4 (&a)[2][4][2], char* unit, char* rawdst, int lane) {
    ExpCode x[2];
#pragma unroll
    for (int f = 0; f < 2; ++f) {
        char* const dst = unit + VAR + f * PL_Y;
        u32x4 pl[6];
        u32 E[8];
#pragma unroll
        for (int q = 0; q < 8; ++q) {
            u32 z[4];
#pragma unroll
            for (int e = 0; e < 4; ++e) { const float y = a[e >> 1][2 * f + (q >> 2)][e & 1][q & 3]; const u32 b = __builtin_bit_cast(u32, y); z[e] = funnel(b, b, 31); }
            pl[q >> 1][2 * (q & 1)] = perm(z[1], z[0], 0x05040100u); pl[q >> 1][2 * (q & 1) + 1] = perm(z[3], z[2], 0x05040100u);
            const u32 h0 = perm(z[1], z[0], 0x07060302u), h1 = perm(z[3], z[2], 0x07060302u);
            pl[4 + (q >> 2)][q & 3] = perm(h1, h0, 0x06040200u); E[q] = perm(h1, h0, 0x07050301u);
        }
        *(volatile LAS u32x4*)(sc + f * 2048 + lane * 32) = (u32x4){E[0], E[1], E[2], E[3]};
        *(volatile LAS u32x4*)(sc + f * 2048 + lane * 32 + 16) = (u32x4){E[4], E[5], E[6], E[7]};
        code_frame<true>(E, lut, x[f], [&](int q) {
            __builtin_amdgcn_sched_barrier(0);
            if (q < 6) st16_sys(dst + q * 1024 + lane * 16, pl[q]);
        });
        st8_sys(dst + 6144 + lane * 8, (u32x2){x[f].cc[0], x[f].cc[1]});
    }
    const u32 raw = put_var<true>(sc, x, unit, lane);
#pragma unroll
    for (int f = 0; f < 2; ++f)
        if ((raw >> f) & 1u) {
#pragma unroll
            for (int q = 0; q < 8; ++q) {
                const int m = 2 * f + (q >> 2), r = q & 3;
                st16_sys(rawdst + f * 8192 + q * 1024 + lane * 16, __builtin_bit_cast(u32x4, ((f32x4){a[0][m][0][r], a[0][m][1][r], a[1][m][0][r], a[1][m][1][r]})));
            }
        }
}
template <class Hook>
__device__ __forceinline__ void unpack_y(const LAS unsigned char* pl, const LAS unsigned char* var, u32 h0, u32 h1, u32 h2, int f, u32 lut, LAS unsigned char* xt,
                                         u32x4 (&o)[8], int lane, Hook mid) {
    const u32x2 k = *(const volatile LAS u32x2*)(pl + 6144 + lane * 8);
    const u32 cc[2] = {k.x, k.y};
    u32 E[8];
    decode_exp<true>(var, h0, h1, h2, f, cc, lut, xt, E, lane);
    mid();
    const u32x4 m0 = *(const volatile LAS u32x4*)(pl + 4096 + lane * 16), m1 = *(const volatile LAS u32x4*)(pl + 5120 + lane * 16);
    const u32 MID[8] = {m0.x, m0.y, m0.z, m0.w, m1.x, m1.y, m1.z, m1.w};
#pragma unroll
    for (int h = 0; h < 4; ++h) {
        const u32x4 lo = *(const volatile LAS u32x4*)(pl + h * 1024 + lane * 16);
#pragma unroll
        for (int j = 0; j < 2; ++j) {
            const int q = 2 * h + j;
            const u32 h0_ = perm(E[q], MID[q], 0x05010400u), h1_ = perm(E[q], MID[q], 0x07030602u);
            const u32 z[4] = {perm(h0_, lo[2 * j], 0x05040100u), perm(h0_, lo[2 * j], 0x07060302u), perm(h1_, lo[2 * j + 1], 0x05040100u), perm(h1_, lo[2 * j + 1], 0x07060302u)};
#pragma unroll
            for (int e4 = 0; e4 < 4; ++e4) o[q][e4] = funnel(z[e4], z[e4], 1);
        }
    }
}

__device__ __forceinline__ void put_block(LAS unsigned char* tb, const u32x4 (&o)[8], float* dst, int lane) {
    const int fr = lane & 15, fq = lane >> 4;
#pragma unroll
    for (int q = 0; q < 8; ++q)
        *(volatile LAS u32x4*)(tb + ((q >> 2) * 16 + fr) * 256 + (((fq * 4 + (q & 3)) ^ (fr & 7)) * 16)) = o[q];
    float* const d = dst + (size_t)fq * NN + fr * 4;
#pragma unroll
    for (int q = 0; q < 8; ++q) {
        f32x4 v = *(volatile LAS f32x4*)(tb + (q * 4 + fq) * 256 + ((fr ^ ((q * 4 + fq) & 7)) * 16));
        v = (f32x4){gelu(v.x), gelu(v.y), gelu(v.z), gelu(v.w)};
        st16_nt((char*)(d + (size_t)q * 4 * NN), __builtin_bit_cast(u32x4, v));
    }
}
__device__ __forceinline__ u32x4 cvt_x(f32x4 a, f32x4 b, int lane) {
    const u32 a0 = pk16(a.x, a.y), a1 = pk16(a.z, a.w), b0 = pk16(b.x, b.y), b1 = pk16(b.z, b.w);
    const bool odd = lane & 1;
    const u32 r0 = (u32)__builtin_amdgcn_mov_dpp((int)(odd ? a0 : b0), 0xB1, 0xf, 0xf, true), r1 = (u32)__builtin_amdgcn_mov_dpp((int)(odd ? a1 : b1), 0xB1, 0xf, 0xf, true);
    return odd ? (u32x4){r0, r1, b0, b1} : (u32x4){a0, a1, r0, r1};
}
__device__ __forceinline__ void put_frame(char* piece, bool isx, int fw, const u32x4 (&c)[4], int lane) {
    if (isx) {
        char* const d = piece + (fw >> 2) * 16384 + (fw & 3) * 512 + ((lane & 15) >> 1) * 2048 + (4 * (lane & 1) + (lane >> 4)) * 16;
#pragma unroll
        for (int q = 0; q < 4; ++q) st16_sys(d + q * 128, c[q]);
    } else {
        char* const d = piece + fw * 2048 + lane * 16;
#pragma unroll
        for (int q = 0; q < 4; ++q) st16_sys(d + (q >> 1) * 16384 + (q & 1) * 1024, c[q]);
    }
}

#define STAMP_AT(i) do {} while (0)

__global__ __launch_bounds__(512, 2) void fused(Params P) {
    extern __shared__ __attribute__((aligned(16))) unsigned char smem[];
    LAS unsigned char* lds = (LAS unsigned char*)smem;
    volatile LAS u32* misc = (volatile LAS u32*)(lds + L_MISC);
    const int tid = threadIdx.x, lane = tid & 63, wid = __builtin_amdgcn_readfirstlane(tid >> 6);
    const int xl = blockIdx.x & 7, jb = blockIdx.x >> 3;
    const int gi = jb - 7;
    const int cw = xl < 4 ? (gi - 14) * 4 + xl : 44 + gi * 4 + (xl - 4);
    int rank = P.rank; u32 first = P.first[(u32)cw < (u32)CRIT ? cw : 0];
    GAS char* wsg = (GAS char*)P.self; const GAS float* pxg = (const GAS float*)P.x; const GAS float* pwg = (const GAS float*)P.w;
    asm volatile("" : "+s"(rank), "+s"(wsg), "+s"(pxg), "+s"(pwg), "+s"(first));
    char* const ws = (char*)wsg; const float* const px = (const float*)pxg; const float* const pw = (const float*)pwg;
    u32* const ctl = (u32*)ws;
    u32* const tmo = ctl + C_TMO;
    LAS unsigned char* const wlds = lds + L_STAGE + wid * 16384;
    const u64 t_in = __builtin_amdgcn_s_memrealtime();
    STAMP_AT(0);
    if (tid == 448) { const u32 c = ld_agt(ctl + C_CALL + 32 * blockIdx.x) + 1u; st_agt(ctl + C_CALL + 32 * blockIdx.x, c); misc[0] = c; }
    if (tid > 0 && tid < 64) misc[tid] = 0u;
    if (tid < 256) ((volatile LAS u32*)(lds + L_SCR))[tid] = 0u;
    if (tid >= 384 && tid < 400) make_lut((LAS u32*)(lds + L_LUT), tid - 384);
    u32 lut = (u32)(size_t)(lds + L_LUT);
    asm volatile("" : "+v"(lut));

    if (jb < 7) {
        LAS unsigned char* const xt = lds + L_XT + wid * 2048;
        LAS unsigned char* const vb = lds + L_VAR + wid * 1040;
        *(volatile LAS u32x4*)(xt + lane * 32) = (u32x4){0, 0, 0, 0}; *(volatile LAS u32x4*)(xt + lane * 32 + 16) = (u32x4){0, 0, 0, 0};
        __syncthreads();
        const u32 call = rfl(misc[0]);
        const int par = call & 1;
        const int peer = (rank + 1 + jb) & 7;
        const char* const pws = P.ws[peer];
        u32* const tick = ctl + C_TICK + 32 * jb;
        const u32 tbase = (call - 1u) * TICKS - TK_0;
        bool ok = true;
        u64 rmask = 0; u32 rbase = 0;
        u32 qline = 0;
        u32* pend = nullptr;

        auto yunit = [&](u32 tk) -> u32 { return (tk < TK_SOLO ? tk : TK_SOLO + ((tk - TK_SOLO) >> 1)) - TK_IN; };
        auto known = [&](u32 tk, int& what) -> bool {
            if (tk < TK_IN) { what = (int)ORDER.item[tk >> 2]; const u32 d = (tk >> 2) - rbase; return d < 64u && ((rmask >> d) & 1ull); }
            const u32 got = (u32)__builtin_amdgcn_readlane((int)qline, (int)(yunit(tk) >> 4));
            what = (int)(got & 31u);
            return got >= (call << 8);
        };
        auto probe = [&](u32 tk) -> const u32* { return tk < TK_IN ? ctl + C_RDY + 320 * peer + (tk >> 2) + lane : ctl + C_YQ + 32 * peer + (lane & 31); };
        auto learn = [&](u32 tk, u32 word) { if (tk < TK_IN) { rbase = tk >> 2; rmask = __ballot(word >= call); } else qline = word; };
        auto await = [&](u32 tk, int& what) -> bool {
            const u64 t0 = __builtin_amdgcn_s_memrealtime();
            for (u32 n = 1; !known(tk, what); ++n) {
                if ((n & 15u) == 0u && (rfl(ld_agt(tmo)) == call || __builtin_amdgcn_s_memrealtime() - t0 > DEADLINE)) { st_agt(tmo, call); return false; }
                if (n > 1) __builtin_amdgcn_s_sleep(2);
                learn(tk, ld_sys(probe(tk)));
            }
            return true;
        };
        auto unit_of = [&](u32 tk, int what) -> size_t {
            if (tk < TK_IN) return (size_t)(((what & 128) ? par : par * 8 + rank) * 128 + (what & 127)) * 4 + (tk & 3u);
            return (size_t)(par * 256 + (2 * rank + (what >> 4)) * 16 + (what & 15)) * 16 + (yunit(tk) & 15u);
        };
        auto packed = [&](u32 tk, int what) -> const char* {
            return pws + (tk < TK_IN ? ((what & 128) ? OFF_EXX : OFF_EXW) + unit_of(tk, what) * UB_IN : OFF_EXY + unit_of(tk, what) * UB_Y) + lane * 16;
        };
#define PULL(q) __builtin_amdgcn_global_load_lds((const GAS u32*)(src + (q) * 1024), (LAS u32*)(buf + (q) * 1024), 16, 0, 17)

        auto first_block = [&](u32 tk, int what) -> const char* { return packed(tk, what) + VAR + (tk >= TK_SOLO && ((tk - TK_SOLO) & 1u) ? PL_Y : 0); };
        auto draw = [&]() -> u32 { u32 d = 0; if (lane == 0) d = add_agt(tick, 1u); return rfl(d) - tbase; };
        auto look = [&](u32 tk) { int w; if (ok && tk < TK_END && !known(tk, w)) learn(tk, ld_sys(probe(tk))); };
        u32 h0 = 0, h1 = 0, h2 = 0;
        auto header = [&](const char* usrc) {
            h0 = ok ? rfl(((volatile LAS u32*)vb)[0]) : 0xC0000000u; h1 = rfl(((volatile LAS u32*)vb)[1]); h2 = rfl(((volatile LAS u32*)vb)[2]);
            if (ok && var_bytes(h0, h1, h2) > (u32)VAR0) {
                if (lane * 16 >= VAR0 && (u32)lane * 16u < var_bytes(h0, h1, h2)) __builtin_amdgcn_global_load_lds((const GAS u32*)usrc, (LAS u32*)vb, 16, 0, 17);
                WAIT_V(0);
            }
        };
#define PULL_VAR() do { if (lane < VAR0 / 16) __builtin_amdgcn_global_load_lds((const GAS u32*)usrc, (LAS u32*)vb, 16, 0, 17); } while (0)

        u32 cur = draw(), nxt = 0;
        while (GATE && __builtin_amdgcn_s_memrealtime() - t_in < (u64)GATE) __builtin_amdgcn_s_sleep(1);
        int what = 0, bsel = 0;
        int have = 0;
        while (cur < TK_IN) {
            LAS unsigned char* const buf = wlds + bsel * 8192;
            if (!have) {
                if (pend) { WAIT_V(0); if (lane == 0) add_agt(pend, 1u); pend = nullptr; }
                if (ok) ok = await(cur, what);
                if (!ok) what = (int)ORDER.item[cur >> 2];
            }
            const char* const usrc = packed(cur, what);
            const char* const src = usrc + VAR;
            for (; ok && have < AHEAD_IN; ++have) { PULL(have); WAIT_V(0); }
            if (ok) PULL(AHEAD_IN);
            nxt = draw();
            WAIT_V(0);
            if (pend && lane == 0) add_agt(pend, 1u);
            pend = nullptr;
            if (ok) {
                PULL(AHEAD_IN + 1); look(nxt); WAIT_V(0);
#pragma unroll
                for (int q = AHEAD_IN + 2; q < 6; ++q) { PULL(q); WAIT_V(0); }
                if (lane < 32) PULL(6);
                PULL_VAR();
                WAIT_V(0);
            }
            header(usrc);
            if (ok && (h0 >> 30)) {
                const char* const src = pws + ((what & 128) ? OFF_RWX : OFF_RWW) + unit_of(cur, what) * 8192 + lane * 16;
                LAS unsigned char* const buf = wlds + (bsel ^ 1) * 8192;
                if (h0 & 0x40000000u) { PULL(0); PULL(1); PULL(2); PULL(3); }
                if (h0 & 0x80000000u) { PULL(4); PULL(5); PULL(6); PULL(7); }
                WAIT_V(0);
            }
            int whatn = 0;
            const bool go = (h0 >> 30) == 0u && nxt < TK_END && known(nxt, whatn);
            const bool isx = what & 128;
            const int t = what & 15, pos = 8 + ((what >> 4) & 7) * 7 + jb;
            const char* const srcn = first_block(nxt, whatn);
            auto ahead = [&](int q) {
                const char* const src = srcn;
                LAS unsigned char* const buf = wlds + (bsel ^ 1) * 8192;
                if (go) PULL(q);
            };
#pragma unroll
            for (int f = 0; f < 2; ++f) {
                if (f) WAIT_V(4);
                ahead(f * (AHEAD_IN / 2));
                u32x4 c[4];
                if ((h0 >> (30 + f)) & 1u) {
#pragma unroll
                    for (int q = 0; q < 4; ++q) c[q] = *(volatile LAS u32x4*)(wlds + (bsel ^ 1) * 8192 + f * 4096 + q * 1024 + lane * 16);
                } else unpack_in(buf + f * PL_IN, vb, h0, h1, h2, f, lut, xt, c, lane, [&] { if (AHEAD_IN == 4) { WAIT_V(0); ahead(2 * f + 1); } });
                put_frame(ws + (isx ? OFF_STA : OFF_STB) + (size_t)(pos * 16 + t) * PIECE, isx, (int)(cur & 3u) * 2 + f, c, lane);
            }
            pend = ctl + (isx ? C_FLAGA : C_FLAGB) + t * 64 + pos;
            WAIT_V(4);
            have = go ? AHEAD_IN : 0;
            cur = nxt; what = whatn; bsel ^= 1;
        }
        STAMP_AT(2);
        int sub = cur >= TK_SOLO ? (int)((cur - TK_SOLO) & 1u) : 0;
        while (cur < TK_END) {
            const bool solo = cur >= TK_SOLO, opens = sub == 0 || solo, closes = sub == 1 || solo;
            LAS unsigned char* const buf = wlds + bsel * 8192;
            if (!have && opens) {
                if (pend) { WAIT_V(0); if (lane == 0) add_agt(pend, 1u); pend = nullptr; }
                if (ok) ok = await(cur, what);
                if (!ok) what = (int)(yunit(cur) >> 4);
            }
            const char* const usrc = packed(cur, what);
            const char* const src = usrc + VAR + (sub ? PL_Y : 0);
            const bool wide = cur >= TK_END - TAIL2;
            int q = have;
            if (ok) { PULL(q); ++q; if (wide) { PULL(q); ++q; } }
            if (opens) nxt = draw();
            if (!opens) look(nxt);
            WAIT_V(0);
            if (pend && lane == 0) add_agt(pend, 1u);
            pend = nullptr;
            if (ok) {
                if (solo) look(nxt);
                for (; q < 6; ++q) { PULL(q); if (!wide) WAIT_V(0); }
                if (lane < 32) PULL(6);
                if (opens) PULL_VAR();
                WAIT_V(0);
            }
            if (opens) header(usrc);
            const bool raw = (h0 >> (30 + sub)) & 1u;
            if (ok && raw) {
                const char* const src = pws + OFF_RWY + (unit_of(cur, what) * 2 + sub) * 8192 + lane * 16;
                PULL(0); PULL(1); PULL(2); PULL(3); PULL(4); PULL(5); PULL(6); PULL(7);
                WAIT_V(0);
            }
            int whatn = what;
            const bool go = ok && (!closes || (nxt < TK_END && known(nxt, whatn)));
            const char* const srcn = closes ? first_block(nxt, whatn) : src + PL_Y;
            int got = 0;
            auto ahead = [&]() {
                const char* const src = srcn;
                LAS unsigned char* const buf = wlds + (bsel ^ 1) * 8192;
                WAIT_V(0);
                if (go) { PULL(got); ++got; }
            };
            ahead();
            u32x4 o[8];
            if (raw) {
#pragma unroll
                for (int q = 0; q < 8; ++q) o[q] = ok ? *(volatile LAS u32x4*)(buf + q * 1024 + lane * 16) : (u32x4){0x7fc00000u, 0x7fc00000u, 0x7fc00000u, 0x7fc00000u};
            } else unpack_y(buf, vb, h0, h1, h2, sub, lut, xt, o, lane, [&] { if (AHEAD_Y >= 2) ahead(); });
            if (AHEAD_Y >= 3) {
#pragma unroll
                for (int q = 0; q < 8; ++q) asm volatile("" : "+v"(o[q]));
                ahead();
            }
            const int F = (int)(yunit(cur) & 15u) * 2 + sub;
            put_block(buf, o, P.out + (size_t)(((what >> 4) & 1) * 256 + ((F >> 1) & 1) * 128 + (F >> 4) * 64 + (F & 1) * 32) * NN + peer * NB + (what & 15) * 256 + ((F >> 2) & 3) * 64, lane);
            WAIT_V(8);
            have = got;
            if (closes) { cur = nxt; what = whatn; sub = cur >= TK_SOLO ? (int)((cur - TK_SOLO) & 1u) : 0; } else sub = 1;
            bsel ^= 1;
        }
        if (pend) { WAIT_V(0); if (lane == 0) add_agt(pend, 1u); }
        STAMP_AT(4);
        return;
    }

    u32 call;
    if (cw < 0) {
        const int g0 = gi * 4 + xl, tj = g0 / 7, dest = (rank + 1 + g0 % 7) & 7, lb = (rank - dest - 1) & 7;
        const float* const src = px + (size_t)(tj * 512 + wid * 32 + (lane >> 4)) * KS + (lane & 15) * 4;
        f32x4 v[8];
#pragma unroll
        for (int i = 0; i < 8; ++i) v[i] = *(const f32x4*)(src + i * 4 * KS);
        char* const dws = P.ws[dest];
        WAIT_L(0); BAR;
        call = rfl(misc[0]);
        const u32x4 c[4] = {cvt_x(v[0], v[1], lane), cvt_x(v[2], v[3], lane), cvt_x(v[4], v[5], lane), cvt_x(v[6], v[7], lane)};
        char* const d = dws + OFF_PSH + (size_t)(((call & 1) * 7 + lb) * 8 + tj) * PIECE + (wid >> 2) * 16384 + (wid & 3) * 512 + ((lane & 15) >> 1) * 2048 + (4 * (lane & 1) + (lane >> 4)) * 16;
#pragma unroll
        for (int q = 0; q < 4; ++q) {
            st16_sys(d + q * 128, c[q]);
            WAIT_V(0);
        }
        WAIT_V(0); BAR;
        if (tid == 0) st_sys((u32*)dws + C_FLAGA + tj * 128 + 8 + lb, 4u * call);
    } else {
        f32x4 v[16];
        const int uj = wid & 3, e_first = (wid >> 2) * NCVT + cw;
        auto decode = [&](int e, bool& isx, int& kk, int& t, int& dest) {
            const int v = (int)(e < CRIT ? first : ORDER.conv[e]);
            isx = v & 128; kk = (v >> 4) & 7; t = v & 15; dest = isx ? rank : ((rank + 1 + (v >> 8)) & 7);
        };
        auto load = [&](int e) {
            bool isx; int kk, t, dest; decode(e, isx, kk, t, dest);
            const char* base = isx ? (const char*)(px + (size_t)(t * 256 + uj * 64 + (lane >> 4)) * KS + kk * 64 + (lane & 15) * 4)
                                   : (const char*)(pw + (size_t)(kk * 64 + uj * 16) * NN + dest * NB + t * 256 + lane * 4);
            const size_t s1 = isx ? 8192 : 131072;
#pragma unroll
            for (int i = 0; i < 16; ++i) v[i] = __builtin_nontemporal_load((const f32x4*)(base + i * s1));
        };
        u32* const peerctl = (u32*)P.ws[(rank + 1 + lane) & 7];
        auto paced_load = [&](int e) {
            while (__builtin_amdgcn_s_memrealtime() - t_in < (u64)((e * PACE) >> 2)) __builtin_amdgcn_s_sleep(1);
            load(e);
        };
        if (e_first < CRIT) load(e_first);
        WAIT_L(0); BAR;
        call = rfl(misc[0]);
        const int par = call & 1;
#pragma unroll 1
        for (int it = 0; it < 4; ++it) {
            const int e = e_first + 2 * NCVT * it;
            if (e >= NCONV) break;
            if (e >= CRIT) paced_load(e);
            bool isx; int kk, t, dest; decode(e, isx, kk, t, dest);
            int eo = (int)ORDER.place[(isx ? 128 : 0) | kk << 4 | t];
            asm volatile("" : "+s"(eo));
            u32x4 c[2][4];
            WAIT_V0_SEEN();
            if (isx) {
#pragma unroll
                for (int j = 0; j < 8; ++j) c[j >> 2][j & 3] = cvt_x(v[2 * j], v[2 * j + 1], lane);
            } else {
#pragma unroll
                for (int f = 0; f < 2; ++f)
#pragma unroll
                    for (int j = 0; j < 4; ++j) c[f][j] = (u32x4){pk16(v[8 * f][j], v[8 * f + 1][j]), pk16(v[8 * f + 2][j], v[8 * f + 3][j]), pk16(v[8 * f + 4][j], v[8 * f + 5][j]), pk16(v[8 * f + 6][j], v[8 * f + 7][j])};
            }
            if (isx ? e >= 8 : dest != rank) {
                const size_t ui = (size_t)((isx ? par : par * 8 + dest) * 128 + kk * 16 + t) * 4 + uj;
                pack_in(wlds, lut, c, ws + (isx ? OFF_EXX : OFF_EXW) + ui * UB_IN, ws + (isx ? OFF_RWX : OFF_RWW) + ui * 8192, lane);
            }
            if (dest == rank) {
                char* const img = ws + (isx ? OFF_STA : OFF_STB) + (size_t)(kk * 16 + t) * PIECE;
                put_frame(img, isx, 2 * uj, c[0], lane);
                put_frame(img, isx, 2 * uj + 1, c[1], lane);
            }
            WAIT_V(0);
            u32 last = 0;
            if (lane == 0) last = __hip_atomic_fetch_add((LAS u32*)(lds + L_MISC) + 8 + it * 2 + (wid >> 2), 1u, __ATOMIC_RELAXED, __HIP_MEMORY_SCOPE_WORKGROUP) == 3u;
            if (rfl(last)) {
                if (isx) { if (lane < 7 && e >= 8) st_sys(peerctl + C_RDY + 320 * rank + eo, call); }
                else if (dest != rank && lane == 0) st_sys((u32*)P.ws[dest] + C_RDY + 320 * rank + eo, call);
                if (dest == rank && lane == 0) st_agt(ctl + (isx ? C_FLAGA : C_FLAGB) + t * 64 + kk, 4u * call);
            }
        }
        __syncthreads();
    }
    STAMP_AT(1);

    const int wr = wid >> 2, wc = wid & 3, fr = lane & 15, fq = lane >> 4;
    const unsigned aoff = fq * 2048 + (wr * 64 + fr) * 16, boff = fq * 2048 + (wc * 16 + fr) * 16;
    const unsigned voff = wid * 1024 + lane * 16, ldsw = wid * 1024;
    const int cg = xl & 3;
    for (int kind = 1; kind <= 2; ++kind) {
        int li, rp, nt;
        if (kind == 1) {
            rp = xl >> 2;
            if (gi < 21) { li = gi % 7; nt = cg * 4 + gi / 7; } else { li = (3 * xl + 3 + (gi - 21)) % 7; nt = cg * 4 + 3; }
        } else {
            if (tid == 0) misc[2] = add_agt(ctl + C_R2 + 32 * cg, 1u) - 50u * (call - 1u);
            __syncthreads();
            const int idx = (int)rfl(misc[2]);
            if ((u32)idx >= 14u) break;
            if (idx < 6) { rp = idx / 3; li = (3 * (cg + 4 * rp) + idx % 3) % 7; nt = cg * 4 + 3; } else { li = 7; rp = (idx - 6) >> 2; nt = cg * 4 + ((idx - 6) & 3); }
        }
        const int dest = (rank + 1 + li) & 7, mt = 2 * dest + rp;
        const char* const cA = ws + OFF_STA + (size_t)mt * PIECE + voff;
        const size_t pshift = OFF_PSH - OFF_STA - (size_t)(64 + mt - (mt >> 1) - (int)(call & 1) * 56) * PIECE;
        const char* const cB = ws + OFF_STB + (size_t)nt * PIECE + voff;
        const u32* const gfa = ctl + C_FLAGA + mt * 64 + lane;
        const u32* const gfb = ctl + C_FLAGB + nt * 64 + lane;
        LAS u32* const scr = (LAS u32*)(lds + L_SCR + (kind == 2 ? 512 : 0));
        u64 known = 0;
        constexpr size_t KSTEP = 16 * PIECE;

#define FETCH_FLAGS() do { __builtin_amdgcn_global_load_lds((const GAS u32*)gfa, scr, 4, 0, 17); __builtin_amdgcn_global_load_lds((const GAS u32*)gfb, scr + 64, 4, 0, 17); } while (0)
#define READ_FLAGS() do { const u32 fa_ = ((volatile LAS u32*)scr)[lane], fb_ = ((volatile LAS u32*)scr)[64 + lane]; known = __ballot(fa_ >= 4u * call && fb_ >= 4u * call); } while (0)
#define NEED(p) do { if (!((known >> (p)) & 1ull)) { \
            const u64 t0_ = __builtin_amdgcn_s_memrealtime(); \
            for (;;) { \
                if (wid == 0) { FETCH_FLAGS(); WAIT_V(0); } \
                READ_FLAGS(); \
                if (((known >> (p)) & 1ull) || rfl(misc[1])) break; \
                if (wid == 0 ? (rfl(ld_agt(tmo)) == call || __builtin_amdgcn_s_memrealtime() - t0_ > DEADLINE) : (__builtin_amdgcn_s_memrealtime() - t0_ > 2 * DEADLINE)) { \
                    st_agt(tmo, call); misc[1] = 1u; break; } \
                __builtin_amdgcn_s_sleep(4); \
            } } } while (0)
#define SA(b, h) (((b) * 2 + (h)) * 16384)
#define SB(b, h) ((4 + (b) * 2 + (h)) * 16384)
#define STAGE(bufoff, g) do { _Pragma("unroll") for (int i_ = 0; i_ < 2; ++i_) \
        __builtin_amdgcn_global_load_lds((const GAS u32*)((g) + i_ * 8192), (LAS u32*)(lds + (bufoff) + ldsw + i_ * 8192), 16, 0, 17); } while (0)
#define LDA(dst, b, h) do { _Pragma("unroll") for (int m = 0; m < 4; ++m) _Pragma("unroll") for (int k = 0; k < 2; ++k) dst[m][k] = *(const LAS h16x8*)(lds + SA(b, h) + aoff + m * 256 + k * 8192); } while (0)
#define LDB(dst, b, h) do { _Pragma("unroll") for (int n = 0; n < 2; ++n) _Pragma("unroll") for (int k = 0; k < 2; ++k) dst[n][k] = *(const LAS h16x8*)(lds + SB(b, h) + boff + n * 1024 + k * 8192); } while (0)
#define MMA(ai, bj, At, Bt) do { __builtin_amdgcn_s_setprio(1); _Pragma("unroll") for (int m = 0; m < 4; ++m) _Pragma("unroll") for (int n = 0; n < 2; ++n) _Pragma("unroll") for (int k = 0; k < 2; ++k) \
        acc[ai][bj][m][n] = __builtin_amdgcn_mfma_f32_16x16x32_f16(Bt[n][k], At[m][k], acc[ai][bj][m][n], 0, 0, 0); __builtin_amdgcn_s_setprio(0); } while (0)

        f32x4 acc[2][2][4][2];
#pragma unroll
        for (int a = 0; a < 2; ++a)
#pragma unroll
            for (int b = 0; b < 2; ++b)
#pragma unroll
                for (int m = 0; m < 4; ++m)
#pragma unroll
                    for (int n = 0; n < 2; ++n) acc[a][b][m][n] = (f32x4){0.f, 0.f, 0.f, 0.f};
        h16x8 At[4][2], B0[2][2], B1[2][2];
        NEED(0);
        STAGE(SB(0, 0), cB); STAGE(SB(0, 1), cB + 16384); STAGE(SA(0, 0), cA); STAGE(SA(0, 1), cA + 16384);
        NEED(1);
        STAGE(SB(1, 0), cB + KSTEP); STAGE(SB(1, 1), cB + KSTEP + 16384); STAGE(SA(1, 0), cA + KSTEP);
        WAIT_V(8);
        if (wr == 1) BAR;
        BAR;
        if (kind == 1) STAMP_AT(2);
        for (int t = 0; t < 64; t += 2) {
            const int p2 = (t + 2) & 63, p3 = p2 + 1;
            auto aoffs = [&](int p) -> size_t { return (rp == 0 && (u32)(p - 8) < 7u) ? pshift + (size_t)p * 8 * PIECE : (size_t)p * KSTEP; };
            const char* a1 = cA + aoffs(t + 1);
            const char* a2 = cA + aoffs(p2); const char* b2 = cB + (size_t)p2 * KSTEP;
            const char* a3 = cA + aoffs(p3); const char* b3 = b2 + KSTEP;
            if (wid == 0 && known != ~0ull) FETCH_FLAGS();
            LDB(B0, 0, 0); LDB(B1, 0, 1); LDA(At, 0, 0); STAGE(SA(1, 1), a1 + 16384);
            WAIT_V(8); WAIT_L(0); BAR; MMA(0, 0, At, B0); MMA(0, 1, At, B1); BAR; SCHED;
            NEED(p2);
            LDA(At, 0, 1); STAGE(SB(0, 0), b2); STAGE(SB(0, 1), b2 + 16384); STAGE(SA(0, 0), a2);
            WAIT_V(8); WAIT_L(0); BAR; MMA(1, 0, At, B0); MMA(1, 1, At, B1); READ_FLAGS(); BAR; SCHED;
            if (wid == 0 && known != ~0ull) FETCH_FLAGS();
            LDB(B0, 1, 0); LDB(B1, 1, 1); LDA(At, 1, 0); STAGE(SA(0, 1), a2 + 16384);
            WAIT_V(8); WAIT_L(0); BAR; MMA(0, 0, At, B0); MMA(0, 1, At, B1); BAR; SCHED;
            NEED(p3);
            LDA(At, 1, 1); STAGE(SB(1, 0), b3); STAGE(SB(1, 1), b3 + 16384); STAGE(SA(1, 0), a3);
            WAIT_V(8); WAIT_L(0); BAR; MMA(1, 0, At, B0); MMA(1, 1, At, B1); READ_FLAGS(); BAR; SCHED;
        }
        WAIT_V(0);
        if (wr == 0) BAR;
        BAR;
        STAMP_AT(1 + 2 * kind);
        if (rfl(misc[1]) != 0u || rfl(ld_agt(tmo)) == call) {
            const float qn = __builtin_nanf("");
#pragma unroll
            for (int a = 0; a < 2; ++a)
#pragma unroll
                for (int b = 0; b < 2; ++b)
#pragma unroll
                    for (int m = 0; m < 4; ++m)
#pragma unroll
                        for (int n = 0; n < 2; ++n) acc[a][b][m][n] = (f32x4){qn, qn, qn, qn};
        }
        int lz = tid; asm volatile("" : "+v"(lz)); lz &= 63;
        if (li == 7) {
            float* const dst = P.out + (size_t)(rp * 256 + wr * 64) * NN + rank * NB + nt * 256 + wc * 64;
#pragma unroll
            for (int f = 0; f < 4; ++f) {
                u32x4 o[8];
#pragma unroll
                for (int q = 0; q < 8; ++q) {
                    const int ai = f >> 1, m = (f & 1) * 2 + (q >> 2), r = q & 3;
                    o[q] = __builtin_bit_cast(u32x4, ((f32x4){acc[ai][0][m][0][r], acc[ai][0][m][1][r], acc[ai][1][m][0][r], acc[ai][1][m][1][r]}));
                }
                put_block(wlds + (f & 1) * 8192, o, dst + (size_t)((f >> 1) * 128 + (f & 1) * 32) * NN, lz);
            }
        } else {
            u32 slot = 0;
            if (tid == 0) slot = add_agt(ctl + C_YSLOT + 32 * dest, 1u) - 32u * (call - 1u);
            const size_t ui = (size_t)((call & 1) * 256 + mt * 16 + nt) * 16 + wid * 2;
            pack_y(wlds, lut, acc[0], ws + OFF_EXY + ui * UB_Y, ws + OFF_RWY + ui * 16384, lz);
            pack_y(wlds + 8192, lut, acc[1], ws + OFF_EXY + (ui + 1) * UB_Y, ws + OFF_RWY + (ui + 1) * 16384, lz);
            WAIT_V(0); BAR;
            if (tid == 0 && slot < 32u) st_sys((u32*)P.ws[dest] + C_YQ + 32 * rank + slot, (call << 8) | (u32)(rp * 16 + nt));
        }
        STAMP_AT(2 + 2 * kind);
    }
    STAMP_AT(7);
}
}

extern "C" void kernel_launch(int rank, int nranks, ncclComm_t comm, hipStream_t stream,
                              void* const* d_in, const int* in_sizes, int n_in,
                              void* d_out, int out_size, const int* in_dtypes, int out_dtype,
                              void* const* d_out_all, void* const* d_ws_all, size_t ws_bytes) {
    using namespace gk;
    (void)comm; (void)in_dtypes; (void)out_dtype; (void)d_out_all;
    if (nranks != NR || (unsigned)rank >= (unsigned)NR || n_in != 2 || ws_bytes < WS_NEED ||
        in_sizes[0] != MM * KS || in_sizes[1] != KS * NN || out_size != KS * NN) {
        fprintf(stderr, "kernel_launch: unexpected problem shape\n");
        return;
    }
    Params p;
    p.x = (const float*)d_in[0]; p.w = (const float*)d_in[1]; p.out = (float*)d_out; p.self = (char*)d_ws_all[rank]; p.rank = rank; p.pad = 0;
    for (int i = 0; i < NR; ++i) p.ws[i] = (char*)d_ws_all[i];
    constexpr Order host_order = make_order();
    for (int i = 0; i < CRIT; ++i) p.first[i] = host_order.conv[i];
    HIP_CHECK(hipFuncSetAttribute((const void*)fused, hipFuncAttributeMaxDynamicSharedMemorySize, LDS_BYTES));
    hipLaunchKernelGGL(fused, dim3(NWG), dim3(512), LDS_BYTES, stream, p);
    HIP_CHECK(hipGetLastError());
}
```
